```python
import numpy as np

kernel_name = "gpu_gemm_rs_m2048_k2048_n2048_bf16_silu"


ABSMAX_RTOL = 0.008

COMPUTE_DTYPE = "bfloat16"
EPILOGUE = "silu"
IN_QUANT = None
OUT_QUANT = None
IN_DTYPES = {"x": "bfloat16", "w_mat": "bfloat16"}
WORLD_SIZE = 8

def _to_bf16(x):
    u = np.ascontiguousarray(x, dtype=np.float32).view(np.uint32)
    return ((u + 0x7FFF + ((u >> 16) & 1)) & 0xFFFF0000).view(np.float32)

def setup_inputs():

    x = _to_bf16(np.random.randn(2048, 2048).astype(np.float32))
    w_mat = _to_bf16((np.random.randn(2048, 2048) / np.sqrt(2048)).astype(np.float32))
    return {"x": x, "w_mat": w_mat}

def _epilogue(y):
    return (y / (1.0 + np.exp(-np.clip(y, -60.0, 60.0)))).astype(np.float32)

def reference(x, w_mat):
    y = (x @ w_mat).astype(np.float32)
    return _epilogue(y)
```

```cpp
#include <hip/hip_runtime.h>
#include <rccl/rccl.h>
#include <cstdio>

typedef __attribute__((ext_vector_type(4))) unsigned u32x4;
typedef __attribute__((ext_vector_type(2))) unsigned u32x2;
typedef __attribute__((ext_vector_type(4))) short s16x4;
typedef __attribute__((ext_vector_type(4))) float f32x4;
typedef __attribute__((ext_vector_type(2))) float f32x2;
typedef __attribute__((ext_vector_type(8))) __bf16 bf16x8;
typedef __attribute__((ext_vector_type(2))) __bf16 bf16x2;
typedef __amdgpu_buffer_rsrc_t rsrc_t;
typedef __attribute__((address_space(3))) unsigned lds_u32;
typedef __attribute__((address_space(3))) s16x4 lds_s16x4;
typedef __attribute__((address_space(3))) void lds_void;

constexpr int NRANK = 8;
constexpr unsigned SLOT = 768u << 10;
constexpr unsigned X_OFF = 0, W_OFF = 256u << 10, C_OFF = 512u << 10;
constexpr unsigned BUF_BYTES = 6 * SLOT + (256u << 10);
constexpr unsigned BUF0 = SLOT + (512u << 10);
constexpr unsigned CTL_OFF = 16u << 20;
static_assert(BUF0 + 3 * BUF_BYTES <= CTL_OFF, "workspace layout");
constexpr unsigned CNT_OFF = CTL_OFF;
constexpr unsigned GATE_OFF = CNT_OFF + 256 * 128;
constexpr unsigned TMO_OFF = GATE_OFF + 8 * 128;
constexpr unsigned PROF_OFF = TMO_OFF + 128;
constexpr unsigned WS_NEED = 17u << 20;
static_assert(PROF_OFF + 256 * 8 * 16 * 4 <= WS_NEED, "workspace layout");
constexpr unsigned IN_BYTES = 1u << 20, OUT_BYTES_ = 1u << 20;
constexpr unsigned long long DEADLINE_TICKS = 50000000ull;
constexpr int SYS = 17;

#define SWIN 4
#define XFIRST 1
#define GATE_N 26
#define CWIN 4
#define CHEAVY 1

struct Args {
  const void* x;
  const void* w;
  void* out;
  void* ws[NRANK];
  int rank;
};

#define STAMP_DECL do {} while (0)
#define STAMP(i) do {} while (0)
#define STAMP_FLUSH() do {} while (0)

__device__ __forceinline__ rsrc_t mk_rsrc(const void* p, unsigned bytes) {
  return __builtin_amdgcn_make_buffer_rsrc((void*)p, 0, bytes, 0x00020000);
}
__device__ __forceinline__ unsigned buf_base(unsigned call) {
  return (call % 3u) * BUF_BYTES;
}
__device__ __forceinline__ bool expired(rsrc_t my, unsigned call, unsigned long long t0) {
  const unsigned tm = __builtin_amdgcn_readfirstlane(__builtin_amdgcn_raw_buffer_load_b32(my, 0, TMO_OFF, SYS));
  if (tm == call || __builtin_amdgcn_s_memrealtime() - t0 > DEADLINE_TICKS) {
    __builtin_amdgcn_raw_buffer_store_b32(call, my, 0, TMO_OFF, SYS);
    return true;
  }
  return false;
}
__device__ __forceinline__ bool wait_words(rsrc_t my, unsigned off, unsigned call, unsigned long long t0) {
  for (unsigned it = 1;; ++it) {
    asm volatile("" ::: "memory");
    const unsigned v = __builtin_amdgcn_raw_buffer_load_b32(my, off, 0, SYS);
    if (__builtin_amdgcn_ballot_w64(v == 0) == 0) break;
    if ((it & 63) == 0 && expired(my, call, t0)) return false;
    __builtin_amdgcn_s_sleep(2);
  }
  asm volatile("" ::: "memory");
  return true;
}
__device__ __forceinline__ u32x4 mark(u32x4 v) {
  v[0] = v[0] ? v[0] : 0x8000u;
  return v;
}
__device__ __forceinline__ unsigned umin(unsigned a, unsigned b) { return a < b ? a : b; }
__device__ __forceinline__ f32x4 mfma(u32x4 a, u32x4 b, f32x4 c) {
  return __builtin_amdgcn_mfma_f32_16x16x32_bf16(__builtin_bit_cast(bf16x8, a), __builtin_bit_cast(bf16x8, b), c, 0, 0, 0);
}
__device__ __forceinline__ unsigned pack2(float lo, float hi) {
  f32x2 v = {lo, hi};
  return __builtin_bit_cast(unsigned, __builtin_convertvector(v, bf16x2));
}
__device__ __forceinline__ u32x4 pack8(f32x4 lo, f32x4 hi) {
  u32x4 r = {pack2(lo[0], lo[1]), pack2(lo[2], lo[3]), pack2(hi[0], hi[1]), pack2(hi[2], hi[3])};
  return r;
}
__device__ __forceinline__ void add8(f32x4& lo, f32x4& hi, u32x4 p) {
  lo[0] += __builtin_bit_cast(float, p[0] << 16); lo[1] += __builtin_bit_cast(float, p[0] & 0xffff0000u);
  lo[2] += __builtin_bit_cast(float, p[1] << 16); lo[3] += __builtin_bit_cast(float, p[1] & 0xffff0000u);
  hi[0] += __builtin_bit_cast(float, p[2] << 16); hi[1] += __builtin_bit_cast(float, p[2] & 0xffff0000u);
  hi[2] += __builtin_bit_cast(float, p[3] << 16); hi[3] += __builtin_bit_cast(float, p[3] & 0xffff0000u);
}
__device__ __forceinline__ float silu(float y) {
  const float z = __builtin_amdgcn_fmed3f(y, -60.0f, 60.0f);
  const float e = __builtin_amdgcn_exp2f(z * -1.4426950408889634f);
  return y * __builtin_amdgcn_rcpf(1.0f + e);
}
__device__ __forceinline__ f32x4 silu4(f32x4 v, float poison) {
  f32x4 r = {silu(v[0] + poison), silu(v[1] + poison), silu(v[2] + poison), silu(v[3] + poison)};
  return r;
}
__device__ __forceinline__ void load_piece(rsrc_t rw, unsigned so, int lane, u32x4* raw) {
  const unsigned vo = (lane >> 2) * 4096 + (lane & 3) * 16;
#pragma unroll
  for (int n = 0; n < 8; ++n) raw[n] = __builtin_amdgcn_raw_buffer_load_b128(rw, vo, so + n * 65536, 0);
}
__device__ __forceinline__ void piece_frags(char* img, int lane, const u32x4* raw, u32x4* f) {
  asm volatile("" ::: "memory");
#pragma unroll
  for (int n = 0; n < 8; ++n) *(u32x4*)(img + n * 1024 + lane * 16) = raw[n];
  asm volatile("" ::: "memory");
  char* rd = img + ((lane >> 4) * 8 + ((lane & 15) >> 2)) * 64 + (lane & 3) * 16;
#pragma unroll
  for (int ks = 0; ks < 4; ++ks)
#pragma unroll
    for (int h = 0; h < 2; ++h) {
      const u32x2 lo = __builtin_bit_cast(u32x2, __builtin_amdgcn_ds_read_tr16_b64_v4i16((lds_s16x4*)(rd + ks * 2048 + h * 8)));
      const u32x2 hi = __builtin_bit_cast(u32x2, __builtin_amdgcn_ds_read_tr16_b64_v4i16((lds_s16x4*)(rd + ks * 2048 + 256 + h * 8)));
      f[2 * ks + h] = u32x4{lo[0], lo[1], hi[0], hi[1]};
    }
}

constexpr unsigned L_X = 0;
constexpr unsigned L_IMG = 96 * 1024;
constexpr unsigned L_CNT = 128 * 1024;

__global__ void __launch_bounds__(512, 2) fused_gemm_rs(Args a) {
  __shared__ __attribute__((aligned(16))) char lds[128 * 1024 + 32];
  const int lane = threadIdx.x & 63;
  const unsigned wave = __builtin_amdgcn_readfirstlane(threadIdx.x >> 6);
  const unsigned s = (unsigned)a.rank & 7u;
  const unsigned e = blockIdx.x & 7u;
  const unsigned idx = blockIdx.x >> 3;
  const unsigned j = lane & 15, g = lane >> 4;
  STAMP_DECL;
  STAMP(0);
  const rsrc_t my = mk_rsrc(a.ws[0], WS_NEED);
  const rsrc_t peer = mk_rsrc(a.ws[e], WS_NEED);
  const rsrc_t rx = mk_rsrc(a.x, IN_BYTES), rw = mk_rsrc(a.w, IN_BYTES), ro = mk_rsrc(a.out, OUT_BYTES_);
  const unsigned cnt_off = CNT_OFF + blockIdx.x * 128 + wave * 4;
  const unsigned cntv = __builtin_amdgcn_raw_buffer_load_b32(my, 0, cnt_off, SYS);
  const unsigned voff_x = (lane >> 2) * 512 + ((lane & 3) ^ ((lane >> 4) & 2)) * 16;
  const unsigned loff_x = j * 64 + (g ^ ((j >> 2) & 2)) * 16;

  if (idx < 16 && wave == 0 && e >= 2) {
    const unsigned sx = idx;
    const unsigned d = s ^ e, da = d >> 2, db = (d >> 1) & 1, dc = d & 1;
    u32x4 v[8][4];
    unsigned cid[8];
    u32x4 raw[2][8];
#pragma unroll
    for (int st = 0; st < 4; ++st) {
      if (st == XFIRST) {
#pragma unroll
        for (int r = 0; r < 2; ++r) {
          const unsigned pu = (sx >> 1) * 4 + 2 * r + (sx & 1);
          load_piece(rw, (128 * dc * 2048 + 1024 * db + 32 * pu) * 2, lane, raw[r]);
          cid[XFIRST + 2 * r] = 64 + pu * 2;
          cid[XFIRST + 2 * r + 1] = 64 + pu * 2 + 1;
        }
      }
      const int slotn = st < XFIRST ? st : st + 4;
      const unsigned o = sx + 16 * st;
      unsigned er, rb;
      if (st < 3) { rb = o / 6; er = 1 + o % 6; er += (er >= e); } else { rb = sx & 7; er = 0; }
      cid[slotn] = er * 8 + rb;
      const unsigned so = ((256 * (d ^ er) + 128 * da + 16 * rb) * 256 + 128 * dc) * 2;
#pragma unroll
      for (int u = 0; u < 4; ++u) v[slotn][u] = __builtin_amdgcn_raw_buffer_load_b128(rx, voff_x, so + u * 64, 0);
    }
    const unsigned call = __builtin_amdgcn_readfirstlane(cntv) + 1;
    const unsigned slot = buf_base(call) + e * SLOT;
    const bool last = sx < 8;
    STAMP(1);
#pragma unroll
    for (int n = 0; n < 32; ++n) {
      if (n == (SWIN < 4 * XFIRST ? SWIN : 4 * XFIRST)) {
        __builtin_amdgcn_sched_barrier(0);
        piece_frags(lds, lane, raw[0], &v[XFIRST][0]);
        piece_frags(lds, lane, raw[1], &v[XFIRST + 2][0]);
        __builtin_amdgcn_sched_barrier(0);
      }
      const u32x4 val = mark(v[n >> 2][n & 3]);
      if (n >= SWIN && n <= GATE_N) asm volatile("s_waitcnt vmcnt(%0)" ::"i"(SWIN - 1) : "memory");
      if (n == GATE_N && lane == 0) __builtin_amdgcn_raw_buffer_store_b32(call, my, 0, GATE_OFF + e * 128 + sx * 4, SYS);
      if (n < 28 || last)
        __builtin_amdgcn_raw_buffer_store_b128(val, peer, lane * 16, slot + cid[n >> 2] * 4096 + (n & 3) * 1024, SYS);
    }
    if (lane == 0) __builtin_amdgcn_raw_buffer_store_b32(call, my, 0, cnt_off, SYS);
    STAMP(2);
    asm volatile("s_waitcnt vmcnt(0)" ::: "memory");
    STAMP(3);
    STAMP_FLUSH();
    return;
  }

  const unsigned long long t0 = __builtin_amdgcn_s_memrealtime();
  const unsigned sa = s >> 2, sb = (s >> 1) & 1;
  const float qnan = __builtin_nanf("");
  const unsigned voff_o = (j * 2048 + 8 * g) * 2;

  if (idx < 16) {
    const unsigned call = __builtin_amdgcn_readfirstlane(cntv) + 1;
    if (e < 2) {
      const unsigned zb = ((call + 2) % 3u) * BUF_BYTES + BUF0 + ((idx * 2 + e) * 8 + wave) * (19u << 10);
      static_assert(256 * (19u << 10) == BUF_BYTES, "clear split");
      const u32x4 zero = {0u, 0u, 0u, 0u};
#pragma unroll
      for (int n = 0; n < 19; ++n) __builtin_amdgcn_raw_buffer_store_b128(zero, my, lane * 16, zb + n * 1024, SYS);
      STAMP(1);
      if (wave < 2) {
        if (lane == 0) __builtin_amdgcn_raw_buffer_store_b32(call, my, 0, cnt_off, SYS);
        STAMP_FLUSH();
        return;
      }
    } else {
      if (wave == 1) return;
      __builtin_amdgcn_s_barrier();
    }
    const unsigned U = blockIdx.x * 6 + (wave - 2);
    const unsigned qd = 1 + U % 3, pu = (U / 3) & 31, rb = (U / 3) >> 5;
    const unsigned fa = sa ^ (qd >> 1), fb = sb ^ (qd & 1);
    u32x4 raw[2][8], xa[8], f[8];
#pragma unroll
    for (int ks = 0; ks < 8; ++ks)
      xa[ks] = __builtin_amdgcn_raw_buffer_load_b128(rx, voff_x, ((256 * s + 128 * fa + 16 * rb) * 256 + 32 * ks) * 2, 0);
#pragma unroll
    for (int r = 0; r < 2; ++r) load_piece(rw, (128 * r * 2048 + 1024 * fb + 32 * pu) * 2, lane, raw[r]);
    __builtin_amdgcn_sched_barrier(0);
    if (lane == 0) __builtin_amdgcn_raw_buffer_store_b32(call, my, 0, cnt_off, SYS);
    char* ximg = lds + 65536 + wave * 8192;
#pragma unroll
    for (int ks = 0; ks < 8; ++ks) *(u32x4*)(ximg + ks * 1024 + lane * 16) = xa[ks];
    asm volatile("" ::: "memory");
#pragma unroll
    for (int ks = 0; ks < 8; ++ks) xa[ks] = *(const u32x4*)(ximg + ks * 1024 + loff_x);
    f32x4 acc[2] = {f32x4{0.f, 0.f, 0.f, 0.f}, f32x4{0.f, 0.f, 0.f, 0.f}};
#pragma unroll
    for (int r = 0; r < 2; ++r) {
      piece_frags(lds + wave * 8192, lane, raw[r], f);
#pragma unroll
      for (int ks = 0; ks < 4; ++ks)
#pragma unroll
        for (int h = 0; h < 2; ++h) acc[h] = mfma(f[2 * ks + h], xa[4 * r + ks], acc[h]);
    }
    const unsigned base = buf_base(call) + 2 * qd * SLOT + C_OFF + ((rb >> 2) * 32 + pu) * 4096 + (rb & 3) * 1024;
    STAMP(3);
    bool ok = true;
    u32x4 p0, p1;
    unsigned tm;
    for (unsigned it = 1;; ++it) {
      asm volatile("" ::: "memory");
      p0 = __builtin_amdgcn_raw_buffer_load_b128(my, lane * 16, base, SYS);
      p1 = __builtin_amdgcn_raw_buffer_load_b128(my, lane * 16, base + SLOT, SYS);
      tm = __builtin_amdgcn_raw_buffer_load_b32(my, 0, TMO_OFF, SYS);
      if (__builtin_amdgcn_ballot_w64(umin(p0[0], p1[0]) == 0) == 0) break;
      if (!ok || ((it & 63) == 0 && expired(my, call, t0))) { ok = false; break; }
      __builtin_amdgcn_s_sleep(2);
    }
    STAMP(4);
    add8(acc[0], acc[1], p0);
    add8(acc[0], acc[1], p1);
    const float poison = (ok && tm != call) ? 0.f : qnan;
    __builtin_amdgcn_raw_buffer_store_b128(pack8(silu4(acc[0], poison), silu4(acc[1], poison)), ro, voff_o,
                                           ((128 * fa + 16 * rb) * 2048 + 1024 * fb + 32 * pu) * 2, 0);
    STAMP(5);
    STAMP_FLUSH();
    return;
  }

  const unsigned p = idx & 7, hb = (idx >> 3) & 1;
  const unsigned nsrc = e >= 2 ? 5 : 6;
  lds_u32* lcnt = (lds_u32*)(lds + L_CNT);
  const unsigned zl = (unsigned)lane < nsrc ? lane : 0;
  const unsigned zslot = (2 + zl + (e >= 2 && 2 + zl >= e)) * SLOT;

  if (wave >= 4) {
    const unsigned q = wave - 4;
#pragma unroll
    for (int ks = 0; ks < 8; ++ks)
      __builtin_amdgcn_raw_ptr_buffer_load_lds(rx, (lds_void*)(lds + L_X + ((ks >> 2) * 4 + q) * 4096 + (ks & 3) * 1024), 16, voff_x,
                                               ((256 * (s ^ e) + 128 * sa + 64 * hb + 16 * q) * 256 + 32 * ks) * 2, 0, 0);
    if (lane == 0) { lcnt[q] = 0; lcnt[4] = 0; }
    const unsigned call = __builtin_amdgcn_readfirstlane(cntv) + 1;
    asm volatile("s_waitcnt vmcnt(0) lgkmcnt(0)" ::: "memory");
    __builtin_amdgcn_s_barrier();
    if (lane == 0) __builtin_amdgcn_raw_buffer_store_b32(call, my, 0, cnt_off, SYS);
    __builtin_amdgcn_s_barrier();
    const unsigned xb = buf_base(call) + X_OFF + (e * 8 + 4 * hb + q) * 4096;
    unsigned pending = (1u << nsrc) - 1, res = 1;
    for (unsigned it = 1; pending; ++it) {
      asm volatile("" ::: "memory");
      const unsigned v = __builtin_amdgcn_raw_buffer_load_b32(my, zslot + 4080, xb, SYS);
      const unsigned ready = (unsigned)__builtin_amdgcn_ballot_w64(v != 0) & pending;
      asm volatile("" ::: "memory");
      if (ready) {
        for (unsigned m = ready; m; m &= m - 1) {
          const unsigned zz = __builtin_ctz(m);
          const unsigned so = xb + (2 + zz + (e >= 2 && 2 + zz >= e)) * SLOT;
          char* dst = lds + L_X + (zz * 4 + q) * 4096;
#pragma unroll
          for (int u = 0; u < 4; ++u)
            __builtin_amdgcn_raw_ptr_buffer_load_lds(my, (lds_void*)(dst + u * 1024), 16, lane * 16, so + u * 1024, 0, SYS);
        }
        asm volatile("s_waitcnt vmcnt(0)" ::: "memory");
        for (unsigned m = ready; m; m &= m - 1) {
          const unsigned zz = __builtin_ctz(m);
          const char* src = lds + L_X + (zz * 4 + q) * 4096 + lane * 16;
          const unsigned mn = umin(umin(*(const unsigned*)src, *(const unsigned*)(src + 1024)),
                                   umin(*(const unsigned*)(src + 2048), *(const unsigned*)(src + 3072)));
          if (__builtin_amdgcn_ballot_w64(mn == 0) == 0) pending &= ~(1u << zz);
        }
      } else {
        __builtin_amdgcn_s_sleep(2);
      }
      if ((it & 63) == 0 && expired(my, call, t0)) { res = 2; break; }
    }
    asm volatile("" ::: "memory");
    if (lane == 0) __hip_atomic_store(lcnt + q, res, __ATOMIC_RELAXED, __HIP_MEMORY_SCOPE_WORKGROUP);
    STAMP(2);
    if (q == 0) {
      if (e >= 2) {
        for (unsigned it = 0; it < 4096; ++it) {
          asm volatile("" ::: "memory");
          const unsigned v = __builtin_amdgcn_raw_buffer_load_b32(my, j * 4, GATE_OFF + e * 128, SYS);
          if (__builtin_amdgcn_ballot_w64(v < call) == 0) break;
          __builtin_amdgcn_s_sleep(2);
        }
      }
      if (lane == 0) __hip_atomic_store(lcnt + 4, 1u, __ATOMIC_RELAXED, __HIP_MEMORY_SCOPE_WORKGROUP);
      STAMP(3);
    }
    STAMP_FLUSH();
    return;
  }

  const unsigned pu = p * 4 + wave;
  f32x4 acc[4][2];
#pragma unroll
  for (int q = 0; q < 4; ++q) acc[q][0] = acc[q][1] = f32x4{0.f, 0.f, 0.f, 0.f};
  {
    u32x4 raw[2][8], f[2][8];
#pragma unroll
    for (int r = 0; r < 2; ++r) load_piece(rw, (128 * r * 2048 + 1024 * sb + 32 * pu) * 2, lane, raw[r]);
    __builtin_amdgcn_sched_barrier(0);
#pragma unroll
    for (int r = 0; r < 2; ++r) piece_frags(lds + L_IMG + wave * 8192, lane, raw[r], f[r]);
    __builtin_amdgcn_s_barrier();
    asm volatile("" ::: "memory");
#pragma unroll
    for (int q = 0; q < 4; ++q) {
      u32x4 xa[8];
#pragma unroll
      for (int ks = 0; ks < 8; ++ks) xa[ks] = *(const u32x4*)(lds + L_X + ((ks >> 2) * 4 + q) * 4096 + (ks & 3) * 1024 + loff_x);
#pragma unroll
      for (int ks = 0; ks < 8; ++ks)
#pragma unroll
        for (int h = 0; h < 2; ++h) acc[q][h] = mfma(f[ks >> 2][2 * (ks & 3) + h], xa[ks], acc[q][h]);
    }
    asm volatile("s_waitcnt lgkmcnt(0)" ::: "memory");
    __builtin_amdgcn_s_barrier();
  }
  const unsigned call = __builtin_amdgcn_readfirstlane(cntv) + 1;
  if (lane == 0) __builtin_amdgcn_raw_buffer_store_b32(call, my, 0, cnt_off, SYS);
  STAMP(1);
  const unsigned bb = buf_base(call);
  u32x4 wf[6][8];
  bool ok = true;
#pragma unroll
  for (int r = 0; r < 2; ++r) {
    const unsigned wb = bb + W_OFF + pu * 8192 + r * 4096;
    ok = wait_words(my, wb + zslot + 4080, call, t0) && ok;
    for (unsigned it = 1;; ++it) {
      asm volatile("" ::: "memory");
      unsigned mn = ~0u;
#pragma unroll
      for (int zz = 0; zz < 6; ++zz) {
        const unsigned so = wb + (2 + zz + (e >= 2 && 2 + zz >= e)) * SLOT;
        if (zz < 5 || e < 2) {
#pragma unroll
          for (int u = 0; u < 4; ++u) wf[zz][4 * r + u] = __builtin_amdgcn_raw_buffer_load_b128(my, lane * 16, so + u * 1024, SYS);
        }
      }
#pragma unroll
      for (int zz = 0; zz < 6; ++zz)
        if (zz < 5 || e < 2) {
#pragma unroll
          for (int u = 0; u < 4; ++u) mn = umin(mn, wf[zz][4 * r + u][0]);
        }
      if (__builtin_amdgcn_ballot_w64(mn == 0) == 0) break;
      if (!ok || ((it & 15) == 0 && expired(my, call, t0))) { ok = false; break; }
      __builtin_amdgcn_s_sleep(2);
    }
    STAMP(2 + r);
  }
  const unsigned cbase = bb + e * SLOT + C_OFF + (hb * 32 + pu) * 4096;
  bool sent = false;
#pragma unroll
  for (int q = 0; q < 4; ++q) {
    unsigned st;
    while ((st = __builtin_amdgcn_readfirstlane(__hip_atomic_load(lcnt + q, __ATOMIC_RELAXED, __HIP_MEMORY_SCOPE_WORKGROUP))) == 0)
      __builtin_amdgcn_s_sleep(1);
    asm volatile("" ::: "memory");
    ok = ok && st == 1;
    STAMP(4 + 2 * q);
    const char* xs = lds + L_X + q * 4096 + loff_x;
    u32x4 xf[4];
#pragma unroll
    for (int n = 0; n < 3; ++n) xf[n] = *(const u32x4*)(xs + n * 1024);
    if (q == 3 && e != 0) {
      sent = __builtin_amdgcn_readfirstlane(__hip_atomic_load(lcnt + 4, __ATOMIC_RELAXED, __HIP_MEMORY_SCOPE_WORKGROUP)) != 0;
      if (sent) {
#pragma unroll
        for (int q2 = 0; q2 < (CWIN < 3 ? CWIN : 3); ++q2)
          __builtin_amdgcn_raw_buffer_store_b128(mark(pack8(acc[q2][0], acc[q2][1])), peer, lane * 16, cbase + q2 * 1024, SYS);
      }
    }
#pragma unroll
    for (int n = 0; n < 20; ++n) {
      xf[(n + 3) & 3] = *(const u32x4*)(xs + ((n + 3) >> 2) * 16384 + ((n + 3) & 3) * 1024);
      __builtin_amdgcn_sched_barrier(0);
      acc[q][0] = mfma(wf[n >> 2][2 * (n & 3)], xf[n & 3], acc[q][0]);
      acc[q][1] = mfma(wf[n >> 2][2 * (n & 3) + 1], xf[n & 3], acc[q][1]);
      __builtin_amdgcn_sched_barrier(0);
    }
    if (e < 2) {
      xf[3] = *(const u32x4*)(xs + 5 * 16384 + 3 * 1024);
#pragma unroll
      for (int n = 20; n < 24; ++n) {
        acc[q][0] = mfma(wf[5][2 * (n & 3)], xf[n & 3], acc[q][0]);
        acc[q][1] = mfma(wf[5][2 * (n & 3) + 1], xf[n & 3], acc[q][1]);
      }
    }
    if (!ok) acc[q][0] = acc[q][1] = f32x4{qnan, qnan, qnan, qnan};
    STAMP(5 + 2 * q);
  }
  if (e != 0) {
    if (!sent) {
      while (__builtin_amdgcn_readfirstlane(__hip_atomic_load(lcnt + 4, __ATOMIC_RELAXED, __HIP_MEMORY_SCOPE_WORKGROUP)) == 0)
        __builtin_amdgcn_s_sleep(1);
      asm volatile("" ::: "memory");
#pragma unroll
      for (int q2 = 0; q2 < (CWIN < 3 ? CWIN : 3); ++q2)
        __builtin_amdgcn_raw_buffer_store_b128(mark(pack8(acc[q2][0], acc[q2][1])), peer, lane * 16, cbase + q2 * 1024, SYS);
    }
#pragma unroll
    for (int q2 = (CWIN < 3 ? CWIN : 3); q2 < 4; ++q2) {
      const u32x4 pk = mark(pack8(acc[q2][0], acc[q2][1]));
      if (CWIN < 4) asm volatile("s_waitcnt vmcnt(%0)" ::"i"(CWIN - 1) : "memory");
      __builtin_amdgcn_raw_buffer_store_b128(pk, peer, lane * 16, cbase + q2 * 1024, SYS);
    }
    STAMP(14);
  } else {
    u32x4 pp[4];
    unsigned tm;
    bool ok2 = true;
    for (unsigned it = 1;; ++it) {
      asm volatile("" ::: "memory");
#pragma unroll
      for (int q = 0; q < 4; ++q) pp[q] = __builtin_amdgcn_raw_buffer_load_b128(my, lane * 16, cbase + SLOT + q * 1024, SYS);
      tm = __builtin_amdgcn_raw_buffer_load_b32(my, 0, TMO_OFF, SYS);
      if (__builtin_amdgcn_ballot_w64(umin(umin(pp[0][0], pp[1][0]), umin(pp[2][0], pp[3][0])) == 0) == 0) break;
      if ((it & 63) == 0 && expired(my, call, t0)) { ok2 = false; break; }
      __builtin_amdgcn_s_sleep(2);
    }
    STAMP(12);
    const float poison = (ok2 && tm != call) ? 0.f : qnan;
#pragma unroll
    for (int q = 0; q < 4; ++q) {
      add8(acc[q][0], acc[q][1], pp[q]);
      __builtin_amdgcn_raw_buffer_store_b128(pack8(silu4(acc[q][0], poison), silu4(acc[q][1], poison)), ro, voff_o,
                                             ((128 * sa + 64 * hb + 16 * q) * 2048 + 1024 * sb + 32 * pu) * 2, 0);
    }
    STAMP(13);
  }
  STAMP_FLUSH();
}

extern "C" void kernel_launch(int rank, int nranks, ncclComm_t comm, hipStream_t stream,
                              void* const* d_in, const int* in_sizes, int n_in,
                              void* d_out, int out_size,
                              const int* in_dtypes, int out_dtype,
                              void* const* d_out_all,
                              void* const* d_ws_all, size_t ws_bytes) {
  if (nranks != NRANK || (unsigned)rank >= (unsigned)NRANK || n_in < 2 || ws_bytes < WS_NEED ||
      in_sizes[0] != 2048 * 256 || in_sizes[1] != 256 * 2048 || out_size != 256 * 2048) {
    fprintf(stderr, "unexpected problem geometry\n");
    return;
  }
  Args a;
  a.x = d_in[0];
  a.w = d_in[1];
  a.out = d_out;
  for (int p = 0; p < NRANK; ++p) a.ws[p] = d_ws_all[p ^ rank];
  a.rank = rank;
  hipLaunchKernelGGL(fused_gemm_rs, dim3(256), dim3(512), 0, stream, a);
  hipError_t err = hipGetLastError();
  if (err != hipSuccess) fprintf(stderr, "HIP error %s at launch\n", hipGetErrorString(err));
}
```

```cpp
#include <hip/hip_runtime.h>
#include <rccl/rccl.h>
#include <cstdio>

typedef __attribute__((ext_vector_type(4))) unsigned u32x4;
typedef __attribute__((ext_vector_type(2))) unsigned u32x2;
typedef __attribute__((ext_vector_type(4))) short s16x4;
typedef __attribute__((ext_vector_type(4))) float f32x4;
typedef __attribute__((ext_vector_type(2))) float f32x2;
typedef __attribute__((ext_vector_type(8))) __bf16 bf16x8;
typedef __attribute__((ext_vector_type(2))) __bf16 bf16x2;
typedef __amdgpu_buffer_rsrc_t rsrc_t;
typedef __attribute__((address_space(3))) unsigned lds_u32;
typedef __attribute__((address_space(3))) s16x4 lds_s16x4;
typedef __attribute__((address_space(3))) void lds_void;

constexpr int NRANK = 8;
constexpr unsigned SLOT = 768u << 10;
constexpr unsigned X_OFF = 0, W_OFF = 256u << 10, C_OFF = 512u << 10;
constexpr unsigned BUF_BYTES = 6 * SLOT + (256u << 10);
constexpr unsigned BUF0 = SLOT + (512u << 10);
constexpr unsigned CTL_OFF = 16u << 20;
static_assert(BUF0 + 3 * BUF_BYTES <= CTL_OFF, "workspace layout");
constexpr unsigned CNT_OFF = CTL_OFF;
constexpr unsigned TMO_OFF = CNT_OFF + 256 * 128;
constexpr unsigned PROF_OFF = TMO_OFF + 128;
constexpr unsigned WS_NEED = 17u << 20;
static_assert(PROF_OFF + 256 * 8 * 16 * 4 <= WS_NEED, "workspace layout");
constexpr unsigned IN_BYTES = 1u << 20, OUT_BYTES_ = 1u << 20;
constexpr unsigned long long DEADLINE_TICKS = 50000000ull;
constexpr int SYS = 17;

#define DT16 115
#define DTC16 106
#define QLEAD 6
#define QLEADC 6
#define C0 105
#define XLAG 10
#define FLAG_A 16
#define FLAG_B 56
#define START_DELAY 12
#define OUTAUX 0
#define CODE_KB 17
#define WARM 2

struct Args {
  const void* x;
  const void* w;
  void* out;
  int rank, pad;
  void* ws[NRANK];
};

#define STAMP_DECL do {} while (0)
#define STAMP(i) do {} while (0)
#define STAMP_FLUSH() do {} while (0)

__device__ __forceinline__ rsrc_t mk_rsrc(const void* p, unsigned bytes) {
  return __builtin_amdgcn_make_buffer_rsrc((void*)p, 0, bytes, 0x00020000);
}
__device__ __forceinline__ unsigned buf_base(unsigned call) {
  return (call % 3u) * BUF_BYTES;
}
__device__ __forceinline__ bool expired(rsrc_t my, unsigned call, unsigned long long t0) {
  const unsigned tm = __builtin_amdgcn_readfirstlane(__builtin_amdgcn_raw_buffer_load_b32(my, 0, TMO_OFF, SYS));
  if (tm == call || __builtin_amdgcn_s_memrealtime() - t0 > DEADLINE_TICKS) {
    __builtin_amdgcn_raw_buffer_store_b32(call, my, 0, TMO_OFF, SYS);
    return true;
  }
  return false;
}
__device__ __forceinline__ bool wait_words(rsrc_t my, unsigned off, unsigned call, unsigned long long t0) {
  for (unsigned it = 1;; ++it) {
    asm volatile("" ::: "memory");
    const unsigned v = __builtin_amdgcn_raw_buffer_load_b32(my, off, 0, SYS);
    if (__builtin_amdgcn_ballot_w64(v == 0) == 0) break;
    if ((it & 63) == 0 && expired(my, call, t0)) return false;
    __builtin_amdgcn_s_sleep(2);
  }
  asm volatile("" ::: "memory");
  return true;
}
__device__ __forceinline__ unsigned now32() { return (unsigned)__builtin_amdgcn_s_memrealtime(); }
__device__ __forceinline__ void wait_until(unsigned t) {
  while ((int)(now32() - t) < 0) __builtin_amdgcn_s_sleep(1);
}
__device__ __forceinline__ u32x4 mark(u32x4 v) {
  v[0] = v[0] ? v[0] : 0x8000u;
  return v;
}
__device__ __forceinline__ unsigned umin(unsigned a, unsigned b) { return a < b ? a : b; }
__device__ __forceinline__ f32x4 mfma(u32x4 a, u32x4 b, f32x4 c) {
  return __builtin_amdgcn_mfma_f32_16x16x32_bf16(__builtin_bit_cast(bf16x8, a), __builtin_bit_cast(bf16x8, b), c, 0, 0, 0);
}
__device__ __forceinline__ unsigned pack2(float lo, float hi) {
  f32x2 v = {lo, hi};
  return __builtin_bit_cast(unsigned, __builtin_convertvector(v, bf16x2));
}
__device__ __forceinline__ u32x4 pack8(f32x4 lo, f32x4 hi) {
  u32x4 r = {pack2(lo[0], lo[1]), pack2(lo[2], lo[3]), pack2(hi[0], hi[1]), pack2(hi[2], hi[3])};
  return r;
}
__device__ __forceinline__ void add8(f32x4& lo, f32x4& hi, u32x4 p) {
  lo[0] += __builtin_bit_cast(float, p[0] << 16); lo[1] += __builtin_bit_cast(float, p[0] & 0xffff0000u);
  lo[2] += __builtin_bit_cast(float, p[1] << 16); lo[3] += __builtin_bit_cast(float, p[1] & 0xffff0000u);
  hi[0] += __builtin_bit_cast(float, p[2] << 16); hi[1] += __builtin_bit_cast(float, p[2] & 0xffff0000u);
  hi[2] += __builtin_bit_cast(float, p[3] << 16); hi[3] += __builtin_bit_cast(float, p[3] & 0xffff0000u);
}
__device__ __forceinline__ float silu(float y) {
  const float z = __builtin_amdgcn_fmed3f(y, -60.0f, 60.0f);
  const float e = __builtin_amdgcn_exp2f(z * -1.4426950408889634f);
  return y * __builtin_amdgcn_rcpf(1.0f + e);
}
__device__ __forceinline__ f32x4 silu4(f32x4 v, float poison) {
  f32x4 r = {silu(v[0] + poison), silu(v[1] + poison), silu(v[2] + poison), silu(v[3] + poison)};
  return r;
}
__device__ __forceinline__ void load_piece(rsrc_t rw, unsigned so, int lane, u32x4* raw) {
  const unsigned vo = (lane >> 2) * 4096 + (lane & 3) * 16;
#pragma unroll
  for (int n = 0; n < 8; ++n) raw[n] = __builtin_amdgcn_raw_buffer_load_b128(rw, vo, so + n * 65536, 0);
}
__device__ __forceinline__ void piece_frags(char* img, int lane, const u32x4* raw, u32x4* f) {
  asm volatile("" ::: "memory");
#pragma unroll
  for (int n = 0; n < 8; ++n) *(u32x4*)(img + n * 1024 + lane * 16) = raw[n];
  asm volatile("" ::: "memory");
  char* rd = img + ((lane >> 4) * 8 + ((lane & 15) >> 2)) * 64 + (lane & 3) * 16;
#pragma unroll
  for (int ks = 0; ks < 4; ++ks)
#pragma unroll
    for (int h = 0; h < 2; ++h) {
      const u32x2 lo = __builtin_bit_cast(u32x2, __builtin_amdgcn_ds_read_tr16_b64_v4i16((lds_s16x4*)(rd + ks * 2048 + h * 8)));
      const u32x2 hi = __builtin_bit_cast(u32x2, __builtin_amdgcn_ds_read_tr16_b64_v4i16((lds_s16x4*)(rd + ks * 2048 + 256 + h * 8)));
      f[2 * ks + h] = u32x4{lo[0], lo[1], hi[0], hi[1]};
    }
}

constexpr unsigned L_X = 0;
constexpr unsigned L_IMG = 96 * 1024;
constexpr unsigned L_CNT = 128 * 1024;

extern "C" __global__ void __launch_bounds__(512, 2) fused_gemm_rs(Args a) {
  __shared__ __attribute__((aligned(16))) char lds[128 * 1024 + 32];
  const unsigned e = blockIdx.x & 7u;
  const void* const ax = a.x; const void* const aw = a.w; void* const aout = a.out;
  void* const ws_my = a.ws[0]; void* const ws_peer = a.ws[e];
  const int arank = a.rank;
  const int lane = threadIdx.x & 63;
  const unsigned wave = __builtin_amdgcn_readfirstlane(threadIdx.x >> 6);
  const unsigned long long t0 = __builtin_amdgcn_s_memrealtime();
  __builtin_amdgcn_sched_barrier(0);
  const unsigned s = (unsigned)arank & 7u;
  const unsigned idx = blockIdx.x >> 3;
  const unsigned j = lane & 15, g = lane >> 4;
  STAMP_DECL;
  STAMP(0);
  const unsigned tref = (unsigned)t0 + C0;
  const rsrc_t my = mk_rsrc(ws_my, WS_NEED);
  const rsrc_t peer = mk_rsrc(ws_peer, WS_NEED);
  const rsrc_t rx = mk_rsrc(ax, IN_BYTES), rw = mk_rsrc(aw, IN_BYTES), ro = mk_rsrc(aout, OUT_BYTES_);
  const unsigned cnt_off = CNT_OFF + blockIdx.x * 128 + wave * 4;
  const unsigned cntv = __builtin_amdgcn_raw_buffer_load_b32(my, 0, cnt_off, SYS);
  const unsigned voff_x = (lane >> 2) * 512 + ((lane & 3) ^ ((lane >> 4) & 2)) * 16;
  const unsigned loff_x = j * 64 + (g ^ ((j >> 2) & 2)) * 16;

  if (__builtin_expect(idx < 16 && wave == 0 && e >= 2, 1)) {
    const unsigned sx = idx;
    const unsigned d = s ^ e, da = d >> 2, db = (d >> 1) & 1, dc = d & 1;
    u32x4 v[8][4];
    unsigned cid[8];
    u32x4 raw[2][8];
#pragma unroll
    for (int st = 0; st < 4; ++st) {
      if (st == 2) {
#pragma unroll
        for (int r = 0; r < 2; ++r) {
          const unsigned pu = (sx >> 1) * 4 + 2 * r + (sx & 1);
          load_piece(rw, (128 * dc * 2048 + 1024 * db + 32 * pu) * 2, lane, raw[r]);
          cid[2 + 2 * r] = 64 + pu * 2;
          cid[3 + 2 * r] = 64 + pu * 2 + 1;
        }
      }
      const int slotn = st < 2 ? st : st + 4;
      unsigned rb, er;
      if (st < 2) { rb = sx & 3; er = 1 + (st == 0 ? sx >> 2 : 4 + ((sx >> 2) & 1)); }
      else { const unsigned o = sx + 16 * st - 8; rb = o / 6; er = 1 + o % 6; }
      er += (er >= e);
      if (st == 3 && sx >= 8) { rb = sx - 8; er = 0; }
      cid[slotn] = er * 8 + rb;
      const unsigned so = ((256 * (d ^ er) + 128 * da + 16 * rb) * 256 + 128 * dc) * 2;
#pragma unroll
      for (int u = 0; u < 4; ++u) v[slotn][u] = __builtin_amdgcn_raw_buffer_load_b128(rx, voff_x + u * 64, so, 0);
      __builtin_amdgcn_sched_barrier(0);
    }
    const unsigned call = __builtin_amdgcn_readfirstlane(cntv) + 1;
    const unsigned slot = buf_base(call) + e * SLOT;
    STAMP(1);
#pragma unroll
    for (int c = 0; c < 8; ++c) {
      if (c == 1) {
        __builtin_amdgcn_sched_barrier(0);
        piece_frags(lds, lane, raw[0], &v[2][0]);
        piece_frags(lds, lane, raw[1], &v[4][0]);
        __builtin_amdgcn_sched_barrier(0);
      }
      if (c != 1 || sx < 8) {
        u32x4 val[4];
#pragma unroll
        for (int u = 0; u < 4; ++u) val[u] = mark(v[c][u]);
        if (c > 0) {
          if (c > 2) asm volatile("s_waitcnt vmcnt(4)" ::: "memory");
          wait_until(tref + ((((c == 1 ? 16 : c * 16 - 8) - QLEAD) + sx) * DT16 >> 4));
        }
#pragma unroll
        for (int u = 0; u < 4; ++u) __builtin_amdgcn_raw_buffer_store_b128(val[u], peer, lane * 16, slot + cid[c] * 4096 + u * 1024, SYS);
      }
      if (c == 0) STAMP(4);
    }
    if (lane == 0) __builtin_amdgcn_raw_buffer_store_b32(call, my, 0, cnt_off, SYS);
    STAMP(2);
    asm volatile("s_waitcnt vmcnt(0)" ::: "memory");
    STAMP(3);
    STAMP_FLUSH();
    return;
  }

  if (__builtin_expect(blockIdx.x < 8 && wave == 1, 0)) {
    unsigned long long code;
    asm volatile("s_getpc_b64 vcc\n\ts_add_u32 vcc_lo, vcc_lo, fused_gemm_rs@rel32@lo+4\n\t"
                 "s_addc_u32 vcc_hi, vcc_hi, fused_gemm_rs@rel32@hi+12\n\ts_mov_b64 %0, vcc" : "=s"(code) : : "vcc", "scc");
    const rsrc_t rc = mk_rsrc((const void*)code, CODE_KB * 1024);
    unsigned t[(CODE_KB + 7) / 8];
#pragma unroll
    for (int n = 0; n < (CODE_KB + 7) / 8; ++n) t[n] = __builtin_amdgcn_raw_buffer_load_b32(rc, lane * 128 + n * 8192, 0, 0);
    {
      const rsrc_t rp = mk_rsrc(ws_peer, WS_NEED);
      const unsigned l6 = lane < 6 ? lane : 0;
      const unsigned off = (l6 >> 1) * BUF_BYTES + (blockIdx.x ? blockIdx.x : 1) * SLOT + (l6 & 1 ? SLOT - 64 : blockIdx.x < 2 ? C_OFF : 0);
      asm volatile("" ::"v"(__builtin_amdgcn_raw_buffer_load_b32(rp, off, 0, SYS)));
    }
#pragma unroll
    for (int n = 0; n < (CODE_KB + 7) / 8; ++n) asm volatile("" ::"v"(t[n]));
  }
  __builtin_amdgcn_s_sleep(START_DELAY);
  const unsigned sa = s >> 2, sb = (s >> 1) & 1;
  const float qnan = __builtin_nanf("");
  const unsigned voff_o = (j * 2048 + 8 * g) * 2;

  if (idx < 16) {
    const unsigned call = __builtin_amdgcn_readfirstlane(cntv) + 1;
    if (e < 2) {
      const unsigned zb = ((call + 2) % 3u) * BUF_BYTES + BUF0 + ((idx * 2 + e) * 8 + wave) * (19u << 10);
      static_assert(256 * (19u << 10) == BUF_BYTES, "clear split");
      const u32x4 zero = {0u, 0u, 0u, 0u};
#pragma unroll
      for (int n = 0; n < 19; ++n) __builtin_amdgcn_raw_buffer_store_b128(zero, my, lane * 16, zb + n * 1024, SYS);
      STAMP(1);
      if (wave < 2) {
        if (lane == 0) __builtin_amdgcn_raw_buffer_store_b32(call, my, 0, cnt_off, SYS);
        STAMP_FLUSH();
        return;
      }
    } else {
      if (wave == 1) return;
      __builtin_amdgcn_s_barrier();
    }
    const unsigned U = blockIdx.x * 6 + (wave - 2);
    const unsigned qd = 1 + U % 3, pu = (U / 3) & 31, rb = (U / 3) >> 5;
    const unsigned fa = sa ^ (qd >> 1), fb = sb ^ (qd & 1);
    u32x4 raw[2][8], xa[8], f[8];
#pragma unroll
    for (int ks = 0; ks < 8; ++ks)
      xa[ks] = __builtin_amdgcn_raw_buffer_load_b128(rx, voff_x, ((256 * s + 128 * fa + 16 * rb) * 256 + 32 * ks) * 2, 0);
#pragma unroll
    for (int r = 0; r < 2; ++r) load_piece(rw, (128 * r * 2048 + 1024 * fb + 32 * pu) * 2, lane, raw[r]);
    const unsigned oso = ((128 * fa + 16 * rb) * 2048 + 1024 * fb + 32 * pu) * 2;
    asm volatile("" ::"v"(__builtin_amdgcn_raw_buffer_load_b32(ro, 0, oso, SYS)));
    __builtin_amdgcn_sched_barrier(0);
    if (lane == 0) __builtin_amdgcn_raw_buffer_store_b32(call, my, 0, cnt_off, SYS);
    char* ximg = lds + 65536 + wave * 8192;
#pragma unroll
    for (int ks = 0; ks < 8; ++ks) *(u32x4*)(ximg + ks * 1024 + lane * 16) = xa[ks];
    asm volatile("" ::: "memory");
#pragma unroll
    for (int ks = 0; ks < 8; ++ks) xa[ks] = *(const u32x4*)(ximg + ks * 1024 + loff_x);
    f32x4 acc[2] = {f32x4{0.f, 0.f, 0.f, 0.f}, f32x4{0.f, 0.f, 0.f, 0.f}};
#pragma unroll
    for (int r = 0; r < 2; ++r) {
      piece_frags(lds + wave * 8192, lane, raw[r], f);
#pragma unroll
      for (int ks = 0; ks < 4; ++ks)
#pragma unroll
        for (int h = 0; h < 2; ++h) acc[h] = mfma(f[2 * ks + h], xa[4 * r + ks], acc[h]);
    }
    const unsigned base = buf_base(call) + 2 * qd * SLOT + C_OFF + ((rb >> 2) * 32 + pu) * 4096 + (rb & 3) * 1024;
    STAMP(3);
    bool ok = true;
    {
      const unsigned k = (rb >> 2) * 32 + (pu & 2) * 8 + (pu >> 2) * 2 + (pu & 1) + FLAG_A;
      wait_until(tref + ((120 * DT16 + (k < FLAG_B ? k : FLAG_B) * DTC16) >> 4));
    }
    u32x4 p0, p1;
    unsigned tm;
    for (unsigned it = 1;; ++it) {
      asm volatile("" ::: "memory");
      p0 = __builtin_amdgcn_raw_buffer_load_b128(my, lane * 16, base, SYS);
      p1 = __builtin_amdgcn_raw_buffer_load_b128(my, lane * 16, base + SLOT, SYS);
      tm = __builtin_amdgcn_raw_buffer_load_b32(my, 0, TMO_OFF, SYS);
      if (__builtin_amdgcn_ballot_w64(umin(p0[0], p1[0]) == 0) == 0) break;
      if (!ok || ((it & 63) == 0 && expired(my, call, t0))) { ok = false; break; }
      __builtin_amdgcn_s_sleep(2);
    }
    STAMP(4);
    add8(acc[0], acc[1], p0);
    add8(acc[0], acc[1], p1);
    const float poison = (ok && tm != call) ? 0.f : qnan;
    __builtin_amdgcn_raw_buffer_store_b128(pack8(silu4(acc[0], poison), silu4(acc[1], poison)), ro, voff_o, oso, OUTAUX);
    STAMP(5);
    STAMP_FLUSH();
    return;
  }

  const unsigned p = idx & 7, hb = (idx >> 3) & 1;
  const unsigned nsrc = e >= 2 ? 5 : 6;
  lds_u32* lcnt = (lds_u32*)(lds + L_CNT);
  const unsigned zl = (unsigned)lane < nsrc ? lane : 0;
  const unsigned zslot = (2 + zl + (e >= 2 && 2 + zl >= e)) * SLOT;

  if (wave >= 4) {
    const unsigned q = wave - 4;
#pragma unroll
    for (int ks = 0; ks < 8; ++ks)
      __builtin_amdgcn_raw_ptr_buffer_load_lds(rx, (lds_void*)(lds + L_X + ((ks >> 2) * 4 + q) * 4096 + (ks & 3) * 1024), 16, voff_x,
                                               ((256 * (s ^ e) + 128 * sa + 64 * hb + 16 * q) * 256 + 32 * ks) * 2, 0, 0);
    if (lane == 0) lcnt[q] = 0;
    const unsigned call = __builtin_amdgcn_readfirstlane(cntv) + 1;
    asm volatile("s_waitcnt vmcnt(0) lgkmcnt(0)" ::: "memory");
    __builtin_amdgcn_s_barrier();
    if (lane == 0) __builtin_amdgcn_raw_buffer_store_b32(call, my, 0, cnt_off, SYS);
    __builtin_amdgcn_s_barrier();
    const unsigned xb = buf_base(call) + X_OFF + (e * 8 + 4 * hb) * 4096 + q * 1024;
    unsigned tb = e == 0 ? 112 + 4 * hb : hb ? 88 + e - 1 - (e >= 3) : 24;
    tb = tref + XLAG + (tb * DT16 >> 4);
    const unsigned tstep = (e == 0 || hb == 0 ? 1 : 6) * DT16 >> 4;
    unsigned so[6];
#pragma unroll
    for (int zz = 0; zz < 6; ++zz) {
      unsigned src = 2 + zz + (e >= 2 && 2 + zz >= e);
      if (zz == 5) src = src > 7 ? (e == 7 ? 6 : 7) : src;
      so[zz] = xb + src * SLOT;
    }
    unsigned todo = 15, res = 1;
    for (unsigned it = 1; todo; ++it) {
      wait_until(tb + __builtin_ctz(todo) * tstep);
      const int el = (int)(now32() - tb);
      const unsigned act = todo & (1u | (el >= (int)tstep) << 1 | (el >= (int)(2 * tstep)) << 2 | (el >= (int)(3 * tstep)) << 3);
      asm volatile("" ::: "memory");
      u32x4 r[4][6];
#pragma unroll
      for (int b = 0; b < 4; ++b)
        if (act >> b & 1) {
#pragma unroll
          for (int zz = 0; zz < 6; ++zz) r[b][zz] = __builtin_amdgcn_raw_buffer_load_b128(my, lane * 16, so[zz] + b * 4096, SYS);
        }
      if ((it & 63) == 0 && expired(my, call, t0)) res = 0x10001;
#pragma unroll
      for (int b = 0; b < 4; ++b)
        if (act >> b & 1) {
          unsigned mn = r[b][0][0];
#pragma unroll
          for (int zz = 1; zz < 6; ++zz) mn = umin(mn, r[b][zz][0]);
          if (__builtin_amdgcn_ballot_w64(mn == 0) == 0 || res != 1) {
#pragma unroll
            for (int zz = 0; zz < 6; ++zz) *(u32x4*)(lds + L_X + (zz * 4 + b) * 4096 + q * 1024 + lane * 16) = r[b][zz];
            asm volatile("s_waitcnt lgkmcnt(0)" ::: "memory");
            if (lane == 0) __hip_atomic_fetch_add(lcnt + b, res, __ATOMIC_RELAXED, __HIP_MEMORY_SCOPE_WORKGROUP);
            todo &= ~(1u << b);
          }
        }
      if (it > 16) __builtin_amdgcn_s_sleep(16);
    }
    STAMP_FLUSH();
    return;
  }

  const unsigned pu = p * 4 + wave;
  f32x4 acc[4][2];
#pragma unroll
  for (int q = 0; q < 4; ++q) acc[q][0] = acc[q][1] = f32x4{0.f, 0.f, 0.f, 0.f};
  {
    u32x4 raw[2][8], f[2][8];
#pragma unroll
    for (int r = 0; r < 2; ++r) load_piece(rw, (128 * r * 2048 + 1024 * sb + 32 * pu) * 2, lane, raw[r]);
    __builtin_amdgcn_sched_barrier(0);
#pragma unroll
    for (int r = 0; r < 2; ++r) piece_frags(lds + L_IMG + wave * 8192, lane, raw[r], f[r]);
    __builtin_amdgcn_s_barrier();
    asm volatile("" ::: "memory");
#pragma unroll
    for (int q = 0; q < 4; ++q) {
      u32x4 xa[8];
#pragma unroll
      for (int ks = 0; ks < 8; ++ks) xa[ks] = *(const u32x4*)(lds + L_X + ((ks >> 2) * 4 + q) * 4096 + (ks & 3) * 1024 + loff_x);
#pragma unroll
      for (int ks = 0; ks < 8; ++ks)
#pragma unroll
        for (int h = 0; h < 2; ++h) acc[q][h] = mfma(f[ks >> 2][2 * (ks & 3) + h], xa[ks], acc[q][h]);
    }
    asm volatile("s_waitcnt lgkmcnt(0)" ::: "memory");
    __builtin_amdgcn_s_barrier();
  }
  const unsigned call = __builtin_amdgcn_readfirstlane(cntv) + 1;
  if (lane == 0) __builtin_amdgcn_raw_buffer_store_b32(call, my, 0, cnt_off, SYS);
  STAMP(1);
  const unsigned bb = buf_base(call);
  u32x4 wf[6][8];
  bool ok = true;
#pragma unroll
  for (int r = 0; r < 2; ++r) {
    const unsigned wb = bb + W_OFF + pu * 8192 + r * 4096;
    ok = wait_words(my, wb + zslot + 4080, call, t0) && ok;
    for (unsigned it = 1;; ++it) {
      asm volatile("" ::: "memory");
      unsigned mn = ~0u;
#pragma unroll
      for (int zz = 0; zz < 6; ++zz) {
        const unsigned so = wb + (2 + zz + (e >= 2 && 2 + zz >= e)) * SLOT;
        if (zz < 5 || e < 2) {
#pragma unroll
          for (int u = 0; u < 4; ++u) wf[zz][4 * r + u] = __builtin_amdgcn_raw_buffer_load_b128(my, lane * 16, so + u * 1024, SYS);
        }
      }
#pragma unroll
      for (int zz = 0; zz < 6; ++zz)
        if (zz < 5 || e < 2) {
#pragma unroll
          for (int u = 0; u < 4; ++u) mn = umin(mn, wf[zz][4 * r + u][0]);
        }
      if (__builtin_amdgcn_ballot_w64(mn == 0) == 0) break;
      if (!ok || ((it & 15) == 0 && expired(my, call, t0))) { ok = false; break; }
      __builtin_amdgcn_s_sleep(2);
    }
    STAMP(2 + r);
  }
  const unsigned cbase = bb + e * SLOT + C_OFF + (hb * 32 + pu) * 4096;
  bool sent = false;
  const unsigned tsend = tref + (e >= 2 ? ((120 - QLEADC) * DT16 + (hb * 32 + (wave >> 1) * 16 + p * 2 + (wave & 1)) * DTC16) >> 4 : 0);
#pragma unroll
  for (int q = 0; q < 4; ++q) {
    unsigned st;
    while (((st = __builtin_amdgcn_readfirstlane(__hip_atomic_load(lcnt + q, __ATOMIC_RELAXED, __HIP_MEMORY_SCOPE_WORKGROUP))) & 0xffff) < 4)
      __builtin_amdgcn_s_sleep(1);
    asm volatile("" ::: "memory");
    ok = ok && st == 4;
    STAMP(4 + 2 * q);
    const char* xs = lds + L_X + q * 4096 + loff_x;
    u32x4 xf[4];
#pragma unroll
    for (int n = 0; n < 3; ++n) xf[n] = *(const u32x4*)(xs + n * 1024);
    if (q == 3 && e != 0) {
      sent = (int)(now32() - tsend) >= 0;
      if (sent) {
#pragma unroll
        for (int q2 = 0; q2 < 3; ++q2)
          __builtin_amdgcn_raw_buffer_store_b128(mark(pack8(acc[q2][0], acc[q2][1])), peer, lane * 16, cbase + q2 * 1024, SYS);
      }
    }
#pragma unroll
    for (int n = 0; n < 20; ++n) {
      xf[(n + 3) & 3] = *(const u32x4*)(xs + ((n + 3) >> 2) * 16384 + ((n + 3) & 3) * 1024);
      __builtin_amdgcn_sched_barrier(0);
      acc[q][0] = mfma(wf[n >> 2][2 * (n & 3)], xf[n & 3], acc[q][0]);
      acc[q][1] = mfma(wf[n >> 2][2 * (n & 3) + 1], xf[n & 3], acc[q][1]);
      __builtin_amdgcn_sched_barrier(0);
    }
    if (e < 2) {
      xf[3] = *(const u32x4*)(xs + 5 * 16384 + 3 * 1024);
#pragma unroll
      for (int n = 20; n < 24; ++n) {
        acc[q][0] = mfma(wf[5][2 * (n & 3)], xf[n & 3], acc[q][0]);
        acc[q][1] = mfma(wf[5][2 * (n & 3) + 1], xf[n & 3], acc[q][1]);
      }
    }
    if (!ok) acc[q][0] = acc[q][1] = f32x4{qnan, qnan, qnan, qnan};
    STAMP(5 + 2 * q);
  }
  if (e != 0) {
    if (!sent) {
      u32x4 pk[3];
#pragma unroll
      for (int q2 = 0; q2 < 3; ++q2) pk[q2] = mark(pack8(acc[q2][0], acc[q2][1]));
      wait_until(tsend);
#pragma unroll
      for (int q2 = 0; q2 < 3; ++q2) __builtin_amdgcn_raw_buffer_store_b128(pk[q2], peer, lane * 16, cbase + q2 * 1024, SYS);
    }
    __builtin_amdgcn_raw_buffer_store_b128(mark(pack8(acc[3][0], acc[3][1])), peer, lane * 16, cbase + 3 * 1024, SYS);
    STAMP(14);
  } else {
    u32x4 pp[4];
    unsigned tm;
    bool ok2 = true;
    for (unsigned it = 1;; ++it) {
      asm volatile("" ::: "memory");
#pragma unroll
      for (int q = 0; q < 4; ++q) pp[q] = __builtin_amdgcn_raw_buffer_load_b128(my, lane * 16, cbase + SLOT + q * 1024, SYS);
      tm = __builtin_amdgcn_raw_buffer_load_b32(my, 0, TMO_OFF, SYS);
      if (__builtin_amdgcn_ballot_w64(umin(umin(pp[0][0], pp[1][0]), umin(pp[2][0], pp[3][0])) == 0) == 0) break;
      if ((it & 63) == 0 && expired(my, call, t0)) { ok2 = false; break; }
      __builtin_amdgcn_s_sleep(2);
    }
    STAMP(12);
    const float poison = (ok2 && tm != call) ? 0.f : qnan;
#pragma unroll
    for (int q = 0; q < 4; ++q) {
      add8(acc[q][0], acc[q][1], pp[q]);
      __builtin_amdgcn_raw_buffer_store_b128(pack8(silu4(acc[q][0], poison), silu4(acc[q][1], poison)), ro, voff_o,
                                             ((128 * sa + 64 * hb + 16 * q) * 2048 + 1024 * sb + 32 * pu) * 2, OUTAUX);
    }
    STAMP(13);
  }
  STAMP_FLUSH();
}

extern "C" void kernel_launch(int rank, int nranks, ncclComm_t comm, hipStream_t stream,
                              void* const* d_in, const int* in_sizes, int n_in,
                              void* d_out, int out_size,
                              const int* in_dtypes, int out_dtype,
                              void* const* d_out_all,
                              void* const* d_ws_all, size_t ws_bytes) {
  if (nranks != NRANK || (unsigned)rank >= (unsigned)NRANK || n_in < 2 || ws_bytes < WS_NEED ||
      in_sizes[0] != 2048 * 256 || in_sizes[1] != 256 * 2048 || out_size != 256 * 2048) {
    fprintf(stderr, "unexpected problem geometry\n");
    return;
  }
  Args a;
  a.x = d_in[0];
  a.w = d_in[1];
  a.out = d_out;
  for (int p = 0; p < NRANK; ++p) a.ws[p] = d_ws_all[p ^ rank];
  a.rank = rank;
  a.pad = 0;
  hipLaunchKernelGGL(fused_gemm_rs, dim3(256), dim3(512), 0, stream, a);
  hipError_t err = hipGetLastError();
  if (err != hipSuccess) fprintf(stderr, "HIP error %s at launch\n", hipGetErrorString(err));
}
```

```cpp
#include <hip/hip_runtime.h>
#include <rccl/rccl.h>
#include <cstdio>

typedef __attribute__((ext_vector_type(4))) unsigned u32x4;
typedef __attribute__((ext_vector_type(2))) unsigned u32x2;
typedef __attribute__((ext_vector_type(4))) short s16x4;
typedef __attribute__((ext_vector_type(4))) float f32x4;
typedef __attribute__((ext_vector_type(2))) float f32x2;
typedef __attribute__((ext_vector_type(8))) __bf16 bf16x8;
typedef __attribute__((ext_vector_type(2))) __bf16 bf16x2;
typedef __amdgpu_buffer_rsrc_t rsrc_t;
typedef __attribute__((address_space(3))) unsigned lds_u32;
typedef __attribute__((address_space(3))) s16x4 lds_s16x4;
typedef __attribute__((address_space(3))) void lds_void;

constexpr int NRANK = 8;
constexpr unsigned SLOT = 768u << 10;
constexpr unsigned X_OFF = 0, W_OFF = 256u << 10, C_OFF = 512u << 10;
constexpr unsigned BUF_BYTES = 6 * SLOT + (256u << 10);
constexpr unsigned BUF0 = SLOT + (512u << 10);
constexpr unsigned CTL_OFF = 16u << 20;
static_assert(BUF0 + 3 * BUF_BYTES <= CTL_OFF, "workspace layout");
constexpr unsigned CNT_OFF = CTL_OFF;
constexpr unsigned TMO_OFF = CNT_OFF + 256 * 128 + 1024;
constexpr unsigned WS_NEED = 17u << 20;
static_assert(TMO_OFF + 128 <= WS_NEED, "workspace layout");
constexpr unsigned IN_BYTES = 1u << 20, OUT_BYTES_ = 1u << 20;
constexpr unsigned long long DEADLINE_TICKS = 50000000ull;
constexpr int SYS = 17;

constexpr int DT16 = 108;
constexpr int DTC16 = 106;
constexpr int C0 = 65;
constexpr int QLEAD = 3;
constexpr int QLEADC = 2;
constexpr int P1START = 96;
constexpr int XLAG = 10;
constexpr int FLAG_A = 6;
constexpr int FLAG_B = 64;
constexpr int START_DELAY = 12;
#define FGAP "4"

#define XCD0 5

struct Group {
  void* ws;
  unsigned slabs;
  unsigned misc;
};
struct Args {
  const void* x;
  const void* w;
  void* out;
  void* ws;
  Group grp[NRANK];
};

__host__ __device__ __forceinline__ unsigned relabel(unsigned r) { return (0x76534210u >> (4 * r)) & 7u; }
static inline unsigned group_peer(unsigned b) { return ((b & 3) << 1) | (b >> 2); }

__device__ __forceinline__ rsrc_t mk_rsrc(const void* p, unsigned bytes) {
  return __builtin_amdgcn_make_buffer_rsrc((void*)p, 0, bytes, 0x00020000);
}
__device__ __forceinline__ void store16(u32x4 v, rsrc_t r, unsigned voff, unsigned soff, const int aux) {
  if (aux) __builtin_amdgcn_raw_buffer_store_b128(v, r, voff, soff, SYS);
  else __builtin_amdgcn_raw_buffer_store_b128(v, r, voff, soff, 0);
  asm volatile("s_nop 1" ::"v"(v) : "memory");
}
__device__ __forceinline__ unsigned buf_base(unsigned call) {
  return (call % 3u) * BUF_BYTES;
}
__device__ __forceinline__ bool expired(rsrc_t my, unsigned call, unsigned long long t0) {
  const unsigned tm = __builtin_amdgcn_readfirstlane(__builtin_amdgcn_raw_buffer_load_b32(my, 0, TMO_OFF, SYS));
  if (tm == call || __builtin_amdgcn_s_memrealtime() - t0 > DEADLINE_TICKS) {
    __builtin_amdgcn_raw_buffer_store_b32(call, my, 0, TMO_OFF, SYS);
    return true;
  }
  return false;
}
__device__ __forceinline__ bool wait_words(rsrc_t my, unsigned off, unsigned call, unsigned long long t0) {
  for (unsigned it = 1;; ++it) {
    asm volatile("" ::: "memory");
    const unsigned v = __builtin_amdgcn_raw_buffer_load_b32(my, off, 0, SYS);
    if (__builtin_amdgcn_ballot_w64(v == 0) == 0) break;
    if ((it & 63) == 0 && expired(my, call, t0)) return false;
    __builtin_amdgcn_s_sleep(2);
  }
  asm volatile("" ::: "memory");
  return true;
}
__device__ __forceinline__ unsigned now32() { return (unsigned)__builtin_amdgcn_s_memrealtime(); }
__device__ __forceinline__ void wait_until(unsigned t) {
  for (int rem; (rem = (int)(t - now32())) > 0;) {
    if (rem > 110) __builtin_amdgcn_s_sleep(24);
    else if (rem > 28) __builtin_amdgcn_s_sleep(5);
    else if (rem > 6) __builtin_amdgcn_s_sleep(1);
  }
}
__device__ __forceinline__ u32x4 mark(u32x4 v) {
  v[0] = v[0] ? v[0] : 0x8000u;
  return v;
}
__device__ __forceinline__ unsigned umin(unsigned a, unsigned b) { return a < b ? a : b; }
__device__ __forceinline__ f32x4 mfma(u32x4 a, u32x4 b, f32x4 c) {
  return __builtin_amdgcn_mfma_f32_16x16x32_bf16(__builtin_bit_cast(bf16x8, a), __builtin_bit_cast(bf16x8, b), c, 0, 0, 0);
}
__device__ __forceinline__ unsigned pack2(float lo, float hi) {
  f32x2 v = {lo, hi};
  return __builtin_bit_cast(unsigned, __builtin_convertvector(v, bf16x2));
}
__device__ __forceinline__ u32x4 pack8(f32x4 lo, f32x4 hi) {
  u32x4 r = {pack2(lo[0], lo[1]), pack2(lo[2], lo[3]), pack2(hi[0], hi[1]), pack2(hi[2], hi[3])};
  return r;
}
__device__ __forceinline__ void add8(f32x4& lo, f32x4& hi, u32x4 p) {
  lo[0] += __builtin_bit_cast(float, p[0] << 16); lo[1] += __builtin_bit_cast(float, p[0] & 0xffff0000u);
  lo[2] += __builtin_bit_cast(float, p[1] << 16); lo[3] += __builtin_bit_cast(float, p[1] & 0xffff0000u);
  hi[0] += __builtin_bit_cast(float, p[2] << 16); hi[1] += __builtin_bit_cast(float, p[2] & 0xffff0000u);
  hi[2] += __builtin_bit_cast(float, p[3] << 16); hi[3] += __builtin_bit_cast(float, p[3] & 0xffff0000u);
}
__device__ __forceinline__ float silu(float y) {
  const float z = __builtin_amdgcn_fmed3f(y, -60.0f, 60.0f);
  const float e = __builtin_amdgcn_exp2f(z * -1.4426950408889634f);
  return y * __builtin_amdgcn_rcpf(1.0f + e);
}
__device__ __forceinline__ f32x4 silu4(f32x4 v, float poison) {
  f32x4 r = {silu(v[0] + poison), silu(v[1] + poison), silu(v[2] + poison), silu(v[3] + poison)};
  return r;
}
__device__ __forceinline__ void load_piece(rsrc_t rw, unsigned so, int lane, u32x4* raw) {
  const unsigned vo = (lane >> 2) * 4096 + (lane & 3) * 16;
#pragma unroll
  for (int n = 0; n < 8; ++n) raw[n] = __builtin_amdgcn_raw_buffer_load_b128(rw, vo, so + n * 65536, 0);
}
__device__ __forceinline__ void piece_frags(char* img, int lane, const u32x4* raw, u32x4* f) {
  asm volatile("" ::: "memory");
#pragma unroll
  for (int n = 0; n < 8; ++n) *(u32x4*)(img + n * 1024 + lane * 16) = raw[n];
  asm volatile("" ::: "memory");
  char* rd = img + ((lane >> 4) * 8 + ((lane & 15) >> 2)) * 64 + (lane & 3) * 16;
#pragma unroll
  for (int ks = 0; ks < 4; ++ks)
#pragma unroll
    for (int h = 0; h < 2; ++h) {
      const u32x2 lo = __builtin_bit_cast(u32x2, __builtin_amdgcn_ds_read_tr16_b64_v4i16((lds_s16x4*)(rd + ks * 2048 + h * 8)));
      const u32x2 hi = __builtin_bit_cast(u32x2, __builtin_amdgcn_ds_read_tr16_b64_v4i16((lds_s16x4*)(rd + ks * 2048 + 256 + h * 8)));
      f[2 * ks + h] = u32x4{lo[0], lo[1], hi[0], hi[1]};
    }
}

constexpr unsigned L_X = 0;
constexpr unsigned L_IMG = 96 * 1024;
constexpr unsigned L_CNT = 128 * 1024;

extern "C" __global__ void __launch_bounds__(512, 2) fused_gemm_rs(Args a) {
  __shared__ __attribute__((aligned(16))) char lds[128 * 1024 + 32];
  const void* const ax = a.x; const void* const aw = a.w; void* const aout = a.out; void* const ws_my = a.ws;
  const Group grp = a.grp[blockIdx.x & 7u];
  __builtin_amdgcn_sched_barrier(0);
  const int lane = threadIdx.x & 63;
  const unsigned wave = __builtin_amdgcn_readfirstlane(threadIdx.x >> 6);
  const unsigned idx = blockIdx.x >> 3;
  const unsigned voff_x = (lane >> 2) * 512 + ((lane & 3) ^ ((lane >> 4) & 2)) * 16;
  unsigned sh0 = 20 + 3 * ((idx >> 2) & 3);
  unsigned voff0 = voff_x + (idx & 3) * 8192;
  asm volatile("" : "+s"(sh0));
  asm volatile("" : "+v"(voff0));
#define tref ((unsigned)t0 + C0)
#define cnt_off (CNT_OFF + (idx * 8 + e) * 128 + wave * 4)
  unsigned smask = idx < 16 && wave == 0 ? 0x60u : 0u;
  asm volatile("" : "+s"(smask));
  __builtin_amdgcn_sched_barrier(0);

  if (__builtin_expect((grp.misc & smask) != 0, 1)) {
    const rsrc_t rx = mk_rsrc(ax, IN_BYTES);
    u32x4 v[8][4];
#pragma unroll
    for (int u = 0; u < 4; ++u)
      v[0][u] = __builtin_amdgcn_raw_buffer_load_b128(rx, voff0 + u * 64, (((grp.misc >> sh0) & 7) << 17) + (grp.misc & 0x10100u), 0);
    __builtin_amdgcn_sched_barrier(0);
    unsigned sx = idx, misc = grp.misc;
    asm volatile("" : "+s"(sx), "+s"(misc));
    const unsigned e = (misc >> 4) & 7;
    const unsigned cntv = __builtin_amdgcn_raw_buffer_load_b32(mk_rsrc(ws_my, WS_NEED), 0, cnt_off, SYS);
    __builtin_amdgcn_sched_barrier(0);
    const unsigned long long t0 = __builtin_amdgcn_s_memrealtime();
    __builtin_amdgcn_sched_barrier(0);
    unsigned cid[8];
    u32x4 raw[2][8];
    auto load_x = [&](const int st) {
      const int slotn = st < 2 ? st : st + 4;
      unsigned rb, er;
      if (st == 0) { rb = sx & 3; er = 1 + (sx >> 2); }
      else if (st == 1) { rb = (sx & 3) + (sx & 8 ? 4 : 0); er = sx & 8 ? 1 : 5 + ((sx >> 2) & 1); }
      else { const unsigned o = sx + 16 * st - 12; rb = o / 5; er = 2 + o % 5; }
      er += (er >= e);
      if (st == 3 && sx >= 4) { rb = (sx - 4) & 7; er = 0; }
      cid[slotn] = er * 8 + rb;
      if (st > 0) {
        const unsigned so = (((grp.slabs >> (4 * er)) & 7) << 17) + (misc & 0x10100u) + rb * 8192;
#pragma unroll
        for (int u = 0; u < 4; ++u) v[slotn][u] = __builtin_amdgcn_raw_buffer_load_b128(rx, voff_x + u * 64, so, 0);
      }
    };
    auto load_w = [&](const int r) {
      const unsigned pu = (sx & 7) * 4 + 2 * r + (sx >> 3);
      const unsigned d = (misc ^ e) & 7;
      load_piece(mk_rsrc(aw, IN_BYTES), (128 * (d & 1) * 2048 + 1024 * ((d >> 1) & 1) + 32 * pu) * 2, lane, raw[r]);
      cid[2 + 2 * r] = 64 + pu * 2;
      cid[3 + 2 * r] = 64 + pu * 2 + 1;
    };
    load_x(0);
    load_x(1);
    load_w(0);
    __builtin_amdgcn_sched_barrier(0);
    const rsrc_t my = mk_rsrc(ws_my, WS_NEED), peer = mk_rsrc(grp.ws, WS_NEED);
    const unsigned call = __builtin_amdgcn_readfirstlane(cntv) + 1;
    const unsigned slot = buf_base(call) + e * SLOT;
#pragma unroll
    for (int c = 0; c < 8; ++c) {
      if (c == 1) {
        __builtin_amdgcn_sched_barrier(0);
        asm volatile("" : "+s"(sx), "+s"(misc));
        load_w(1);
        load_x(2);
        load_x(3);
        piece_frags(lds, lane, raw[0], &v[2][0]);
        __builtin_amdgcn_sched_barrier(0);
      }
      if (c == 3) {
        __builtin_amdgcn_sched_barrier(0);
        piece_frags(lds, lane, raw[1], &v[4][0]);
        __builtin_amdgcn_sched_barrier(0);
      }
      if ((c != 1 && c != 7) || sx < 12) {
        u32x4 val[4];
#pragma unroll
        for (int u = 0; u < 4; ++u) val[u] = mark(v[c][u]);
        const unsigned pos = (((c == 0 ? 0 : c == 1 ? 16 : c * 16 - 4) + sx) * DT16) >> 4;
        if (c > 0) {
          asm volatile("" : "+v"(val[0]), "+v"(val[1]), "+v"(val[2]), "+v"(val[3]));
          if (c > 2) asm volatile("s_waitcnt vmcnt(4)" ::: "memory");
          wait_until(tref - (QLEAD * DT16 >> 4) + pos);
        }
#pragma unroll
        for (int u = 0; u < 4; ++u) store16(val[u], peer, lane * 16 + u * 1024, slot + cid[c] * 4096, SYS);
      }
    }
    if (lane == 0) __builtin_amdgcn_raw_buffer_store_b32(call, my, 0, cnt_off, SYS);
    asm volatile("s_waitcnt vmcnt(0)" ::: "memory");
    return;
  }

  const unsigned e = (grp.misc >> 4) & 7;
  const rsrc_t my = mk_rsrc(ws_my, WS_NEED), peer = mk_rsrc(grp.ws, WS_NEED);
  const rsrc_t rx = mk_rsrc(ax, IN_BYTES), rw = mk_rsrc(aw, IN_BYTES), ro = mk_rsrc(aout, OUT_BYTES_);
  const unsigned cntv = __builtin_amdgcn_raw_buffer_load_b32(my, 0, cnt_off, SYS);
  __builtin_amdgcn_sched_barrier(0);
  const unsigned long long t0 = __builtin_amdgcn_s_memrealtime();
  __builtin_amdgcn_sched_barrier(0);
  const unsigned s = grp.misc & 7u;
  const unsigned j = lane & 15, g = lane >> 4;
  const unsigned loff_x = j * 64 + (g ^ ((j >> 2) & 2)) * 16;

  if (__builtin_expect(idx == 15 && wave == 1, 0)) {
    unsigned long long code, here;
    asm volatile("s_getpc_b64 %1\n\ts_getpc_b64 vcc\n\ts_add_u32 vcc_lo, vcc_lo, fused_gemm_rs@rel32@lo+4\n\t"
                 "s_addc_u32 vcc_hi, vcc_hi, fused_gemm_rs@rel32@hi+12\n\ts_mov_b64 %0, vcc"
                 : "=s"(code), "=&s"(here) : : "vcc", "scc");
    const unsigned len = (unsigned)(here - code);
    const rsrc_t rc = mk_rsrc((const void*)code, len);
    unsigned t[3];
#pragma unroll
    for (int n = 0; n < 3; ++n) t[n] = __builtin_amdgcn_raw_buffer_load_b32(rc, lane * 128 + n * 8192, 0, 0);
    const unsigned l6 = lane < 6 ? lane : 0;
    const unsigned t3 = __builtin_amdgcn_raw_buffer_load_b32(peer, (l6 >> 1) * BUF_BYTES + (e ? e : 1) * SLOT + (l6 & 1 ? SLOT - 64 : e < 2 ? C_OFF : 0), 0, SYS);
    const unsigned t4 = __builtin_amdgcn_raw_buffer_load_b32(ro, (lane & 1) * (OUT_BYTES_ - 64), 0, SYS);
    const unsigned t5 = __builtin_amdgcn_raw_buffer_load_b32(my, (lane & 15) << 20, 0, SYS);
    asm volatile("" ::"v"(t[0]), "v"(t[1]), "v"(t[2]), "v"(t3), "v"(t4), "v"(t5));
  }
  __builtin_amdgcn_s_sleep(START_DELAY);
  const unsigned sa = s >> 2, sb = (s >> 1) & 1;
  const float qnan = __builtin_nanf("");
  const unsigned voff_o = (j * 2048 + 8 * g) * 2;

  if (idx < 16) {
    const unsigned call = __builtin_amdgcn_readfirstlane(cntv) + 1;
    if (e < 2) {
      const unsigned zb = ((call + 2) % 3u) * BUF_BYTES + BUF0 + ((idx * 2 + e) * 8 + wave) * (19u << 10);
      static_assert(256 * (19u << 10) == BUF_BYTES, "clear split");
      const u32x4 zero = {0u, 0u, 0u, 0u};
#pragma unroll
      for (int n = 0; n < 19; ++n) store16(zero, my, lane * 16, zb + n * 1024, SYS);
      if (wave < 2) {
        if (lane == 0) __builtin_amdgcn_raw_buffer_store_b32(call, my, 0, cnt_off, SYS);
        asm volatile("s_waitcnt vmcnt(0)" ::: "memory");
        return;
      }
    } else {
      if (wave == 1) return;
      __builtin_amdgcn_s_barrier();
    }
    const unsigned U = (wave - 2) * 128 + blockIdx.x;
    const unsigned fk = U / 12, fr = U % 12;
    const unsigned qd = 1 + (fr >> 2), rb = (fk >> 5) * 4 + (fr & 3);
    const unsigned pu = (fk & 7) * 4 + ((fk >> 3) & 3);
    const unsigned fa = sa ^ (qd >> 1), fb = sb ^ (qd & 1);
    u32x4 raw[2][8], xa[8], f[8];
#pragma unroll
    for (int ks = 0; ks < 8; ++ks)
      xa[ks] = __builtin_amdgcn_raw_buffer_load_b128(rx, voff_x, ((256 * relabel(s) + 128 * fa + 16 * rb) * 256 + 32 * ks) * 2, 0);
#pragma unroll
    for (int r = 0; r < 2; ++r) load_piece(rw, (128 * r * 2048 + 1024 * fb + 32 * pu) * 2, lane, raw[r]);
    const unsigned oso = ((128 * fa + 16 * rb) * 2048 + 1024 * fb + 32 * pu) * 2;
    __builtin_amdgcn_sched_barrier(0);
    if (lane == 0) __builtin_amdgcn_raw_buffer_store_b32(call, my, 0, cnt_off, SYS);
    char* ximg = lds + 65536 + wave * 8192;
#pragma unroll
    for (int ks = 0; ks < 8; ++ks) *(u32x4*)(ximg + ks * 1024 + lane * 16) = xa[ks];
    asm volatile("" ::: "memory");
#pragma unroll
    for (int ks = 0; ks < 8; ++ks) xa[ks] = *(const u32x4*)(ximg + ks * 1024 + loff_x);
    f32x4 acc[2] = {f32x4{0.f, 0.f, 0.f, 0.f}, f32x4{0.f, 0.f, 0.f, 0.f}};
#pragma unroll
    for (int r = 0; r < 2; ++r) {
      piece_frags(lds + wave * 8192, lane, raw[r], f);
#pragma unroll
      for (int ks = 0; ks < 4; ++ks)
#pragma unroll
        for (int h = 0; h < 2; ++h) acc[h] = mfma(f[2 * ks + h], xa[4 * r + ks], acc[h]);
    }
    const unsigned base = buf_base(call) + 2 * qd * SLOT + C_OFF + ((rb >> 2) * 32 + pu) * 4096 + (rb & 3) * 1024;
    unsigned tm = __builtin_amdgcn_raw_buffer_load_b32(my, 0, TMO_OFF, SYS);
    asm volatile("" : "+v"(tm));
    bool ok = true;
    {
      const unsigned k = fk + FLAG_A;
      wait_until(tref + ((120 * DT16 + (k < FLAG_B ? k : FLAG_B) * DTC16) >> 4));
    }
    u32x4 p0, p1;
    {
      const u32x4 rs = {(unsigned)(size_t)ws_my, (unsigned)((size_t)ws_my >> 32) & 0xffffu, WS_NEED, 0x00020000u};
      const unsigned vo = lane * 16, b1 = base + SLOT;
      for (;;) {
        unsigned got, cnt = 32;
#define P_LD(R0, R1) "buffer_load_dwordx4 v[" R0 "], %10, %11, %12 offen sc0 sc1\n\tbuffer_load_dwordx4 v[" R1 "], %10, %11, %13 offen sc0 sc1\n\t"
#define P_CK(A, B, L) "s_waitcnt vmcnt(4)\n\tv_min_u32 %0, v" A ", v" B "\n\tv_cmp_eq_u32 vcc, 0, %0\n\ts_cbranch_vccz " L "%=\n\ts_sleep " FGAP "\n\t"
#define P_MV(A0, A1, A2, A3, B0, B1, B2, B3) "v_mov_b32 %0, v" A0 "\n\tv_mov_b32 %1, v" A1 "\n\tv_mov_b32 %2, v" A2 "\n\tv_mov_b32 %3, v" A3 "\n\t" \
          "v_mov_b32 %4, v" B0 "\n\tv_mov_b32 %5, v" B1 "\n\tv_mov_b32 %6, v" B2 "\n\tv_mov_b32 %7, v" B3 "\n\ts_mov_b32 %8, 1\n\t"
        asm volatile(
            "s_nop 4\n\t"
            P_LD("224:227", "228:231") "s_sleep " FGAP "\n\t"
            P_LD("232:235", "236:239") "s_sleep " FGAP "\n\t"
            ".Lpl%=:\n\t"
            P_LD("240:243", "244:247") P_CK("224", "228", ".Lpa")
            P_LD("224:227", "228:231") P_CK("232", "236", ".Lpb")
            P_LD("232:235", "236:239") P_CK("240", "244", ".Lpc")
            "s_sub_u32 %9, %9, 1\n\ts_cbranch_scc0 .Lpl%=\n\t"
            "s_waitcnt vmcnt(0)\n\t"
            P_MV("224", "225", "226", "227", "228", "229", "230", "231") "s_mov_b32 %8, 0\n\ts_branch .Lpe%=\n\t"
            ".Lpa%=:\n\t" P_MV("224", "225", "226", "227", "228", "229", "230", "231") "s_branch .Lpe%=\n\t"
            ".Lpb%=:\n\t" P_MV("232", "233", "234", "235", "236", "237", "238", "239") "s_branch .Lpe%=\n\t"
            ".Lpc%=:\n\t" P_MV("240", "241", "242", "243", "244", "245", "246", "247")
            ".Lpe%=:"
            : "=&v"(p0[0]), "=&v"(p0[1]), "=&v"(p0[2]), "=&v"(p0[3]), "=&v"(p1[0]), "=&v"(p1[1]), "=&v"(p1[2]), "=&v"(p1[3]),
              "=&s"(got), "+s"(cnt)
            : "v"(vo), "s"(rs), "s"(base), "s"(b1)
            : "memory", "vcc", "scc", "v224", "v225", "v226", "v227", "v228", "v229", "v230", "v231", "v232", "v233", "v234", "v235",
              "v236", "v237", "v238", "v239", "v240", "v241", "v242", "v243", "v244", "v245", "v246", "v247");
#undef P_LD
#undef P_CK
#undef P_MV
        if (got) break;
        if (expired(my, call, t0)) { ok = false; break; }
      }
    }
    add8(acc[0], acc[1], p0);
    add8(acc[0], acc[1], p1);
    const float poison = (ok && tm != call) ? 0.f : qnan;
    store16(pack8(silu4(acc[0], poison), silu4(acc[1], poison)), ro, voff_o, oso, 0);
    return;
  }

  const unsigned p = idx & 7, hb = (idx >> 3) & 1;
  const unsigned nsrc = e >= 2 ? 5 : 6;
  lds_u32* lcnt = (lds_u32*)(lds + L_CNT);
  const unsigned zl = (unsigned)lane < nsrc ? lane : 0;
  const unsigned zslot = (2 + zl + (e >= 2 && 2 + zl >= e)) * SLOT;

  if (wave >= 4) {
    const unsigned q = wave - 4;
#pragma unroll
    for (int ks = 0; ks < 8; ++ks)
      __builtin_amdgcn_raw_ptr_buffer_load_lds(rx, (lds_void*)(lds + L_X + ((ks >> 2) * 4 + q) * 4096 + (ks & 3) * 1024), 16, voff_x,
                                               ((256 * relabel(s ^ e) + 128 * sa + 64 * hb + 16 * q) * 256 + 32 * ks) * 2, 0, 0);
    if (lane == 0) lcnt[q] = 0;
    const unsigned call = __builtin_amdgcn_readfirstlane(cntv) + 1;
    asm volatile("s_waitcnt vmcnt(0) lgkmcnt(0)" ::: "memory");
    __builtin_amdgcn_s_barrier();
    if (lane == 0) __builtin_amdgcn_raw_buffer_store_b32(call, my, 0, cnt_off, SYS);
    __builtin_amdgcn_s_barrier();
    const unsigned xb = buf_base(call) + X_OFF + (e * 8 + 4 * hb) * 4096 + (q & 1) * 2048;
    unsigned tb = e == 0 ? 112 + 4 * hb : hb == 0 || e == 1 ? 28 : 92 + e - 2 - (e >= 3);
    const unsigned tstep = (e < 2 || hb == 0 ? 1 : 5) * DT16 >> 4;
    tb = tref + XLAG + (tb * DT16 >> 4);
    unsigned so[6];
#pragma unroll
    for (int zz = 0; zz < 6; ++zz) {
      so[zz] = xb + (2 + zz + (e >= 2 && 2 + zz >= e)) * SLOT;
    }
    unsigned res = 2;
#pragma unroll 1
    for (unsigned b = q >> 1; b < 4; b += 2) {
      wait_until(tb + b * tstep);
      u32x4 r[6][2];
      for (unsigned it = 1;; ++it) {
        asm volatile("" ::: "memory");
#pragma unroll
        for (int zz = 0; zz < 6; ++zz)
          if (zz < 5 || e < 2) {
#pragma unroll
            for (int h = 0; h < 2; ++h) r[zz][h] = __builtin_amdgcn_raw_buffer_load_b128(my, lane * 16 + h * 1024, so[zz] + b * 4096, SYS);
          }
        unsigned mn = ~0u;
#pragma unroll
        for (int zz = 0; zz < 6; ++zz)
          if (zz < 5 || e < 2) mn = umin(mn, umin(r[zz][0][0], r[zz][1][0]));
        if (__builtin_amdgcn_ballot_w64(mn == 0) == 0) break;
        if (res != 2 || ((it & 63) == 0 && expired(my, call, t0))) { res = 0x10002; break; }
        if (!wait_words(my, xb + zslot + b * 4096 + 2032, call, t0)) res = 0x10002;
      }
      char* dst = lds + L_X + b * 4096 + (q & 1) * 2048 + lane * 16;
#pragma unroll
      for (int zz = 0; zz < 6; ++zz)
        if (zz < 5 || e < 2) {
#pragma unroll
          for (int h = 0; h < 2; ++h) *(u32x4*)(dst + zz * 16384 + h * 1024) = r[zz][h];
        }
      asm volatile("s_waitcnt lgkmcnt(0)" ::: "memory");
      if (lane == 0) __hip_atomic_fetch_add(lcnt + b, res, __ATOMIC_RELAXED, __HIP_MEMORY_SCOPE_WORKGROUP);
    }
    return;
  }

  const unsigned pu = p * 4 + wave;
  f32x4 acc[4][2];
#pragma unroll
  for (int q = 0; q < 4; ++q) acc[q][0] = acc[q][1] = f32x4{0.f, 0.f, 0.f, 0.f};
  {
    u32x4 raw[2][8], f[2][8];
#pragma unroll
    for (int r = 0; r < 2; ++r) load_piece(rw, (128 * r * 2048 + 1024 * sb + 32 * pu) * 2, lane, raw[r]);
    __builtin_amdgcn_sched_barrier(0);
#pragma unroll
    for (int r = 0; r < 2; ++r) piece_frags(lds + L_IMG + wave * 8192, lane, raw[r], f[r]);
    __builtin_amdgcn_s_barrier();
    asm volatile("" ::: "memory");
#pragma unroll
    for (int q = 0; q < 4; ++q) {
      u32x4 xa[8];
#pragma unroll
      for (int ks = 0; ks < 8; ++ks) xa[ks] = *(const u32x4*)(lds + L_X + ((ks >> 2) * 4 + q) * 4096 + (ks & 3) * 1024 + loff_x);
#pragma unroll
      for (int ks = 0; ks < 8; ++ks)
#pragma unroll
        for (int h = 0; h < 2; ++h) acc[q][h] = mfma(f[ks >> 2][2 * (ks & 3) + h], xa[ks], acc[q][h]);
    }
    asm volatile("s_waitcnt lgkmcnt(0)" ::: "memory");
    __builtin_amdgcn_s_barrier();
  }
  const unsigned call = __builtin_amdgcn_readfirstlane(cntv) + 1;
  if (lane == 0) __builtin_amdgcn_raw_buffer_store_b32(call, my, 0, cnt_off, SYS);
  const unsigned bb = buf_base(call);
  u32x4 wf[6][8];
  bool ok = true;
#pragma unroll
  for (int r = 0; r < 2; ++r) {
    const unsigned wb = bb + W_OFF + pu * 8192 + r * 4096;
    ok = wait_words(my, wb + zslot + 4080, call, t0) && ok;
    for (unsigned it = 1;; ++it) {
      asm volatile("" ::: "memory");
      unsigned mn = ~0u;
#pragma unroll
      for (int zz = 0; zz < 6; ++zz) {
        const unsigned so = wb + (2 + zz + (e >= 2 && 2 + zz >= e)) * SLOT;
        if (zz < 5 || e < 2) {
#pragma unroll
          for (int u = 0; u < 4; ++u) wf[zz][4 * r + u] = __builtin_amdgcn_raw_buffer_load_b128(my, lane * 16, so + u * 1024, SYS);
        }
      }
#pragma unroll
      for (int zz = 0; zz < 6; ++zz)
        if (zz < 5 || e < 2) {
#pragma unroll
          for (int u = 0; u < 4; ++u) mn = umin(mn, wf[zz][4 * r + u][0]);
        }
      if (__builtin_amdgcn_ballot_w64(mn == 0) == 0) break;
      if (!ok || ((it & 15) == 0 && expired(my, call, t0))) { ok = false; break; }
      __builtin_amdgcn_s_sleep(2);
    }
  }
  const unsigned cbase = bb + e * SLOT + C_OFF + (hb * 32 + pu) * 4096;
  bool sent = false;
  const unsigned kslot = (hb * 32 + wave * 8 + p) * DTC16 >> 4;
  const unsigned tsend = tref + (e >= 2 ? ((120 - QLEADC) * DT16 >> 4) + kslot
                                        : (P1START * DT16 + (wave * 16 + p * 2 + hb) * DTC16) >> 4);
#pragma unroll
  for (int q = 0; q < 4; ++q) {
    unsigned st;
    while (((st = __builtin_amdgcn_readfirstlane(__hip_atomic_load(lcnt + q, __ATOMIC_RELAXED, __HIP_MEMORY_SCOPE_WORKGROUP))) & 0xffff) < 4)
      __builtin_amdgcn_s_sleep(1);
    asm volatile("" ::: "memory");
    ok = ok && st == 4;
    const char* xs = lds + L_X + q * 4096 + loff_x;
    u32x4 xf[4];
#pragma unroll
    for (int n = 0; n < 3; ++n) xf[n] = *(const u32x4*)(xs + n * 1024);
    if (q == 3 && e != 0) {
      sent = (int)(now32() - tsend) >= 0;
      if (sent) {
#pragma unroll
        for (int q2 = 0; q2 < 3; ++q2)
          store16(mark(pack8(acc[q2][0], acc[q2][1])), peer, lane * 16, cbase + q2 * 1024, SYS);
      }
    }
#pragma unroll
    for (int n = 0; n < 20; ++n) {
      xf[(n + 3) & 3] = *(const u32x4*)(xs + ((n + 3) >> 2) * 16384 + ((n + 3) & 3) * 1024);
      __builtin_amdgcn_sched_barrier(0);
      acc[q][0] = mfma(wf[n >> 2][2 * (n & 3)], xf[n & 3], acc[q][0]);
      acc[q][1] = mfma(wf[n >> 2][2 * (n & 3) + 1], xf[n & 3], acc[q][1]);
      __builtin_amdgcn_sched_barrier(0);
    }
    if (e < 2) {
      xf[3] = *(const u32x4*)(xs + 5 * 16384 + 3 * 1024);
#pragma unroll
      for (int n = 20; n < 24; ++n) {
        acc[q][0] = mfma(wf[5][2 * (n & 3)], xf[n & 3], acc[q][0]);
        acc[q][1] = mfma(wf[5][2 * (n & 3) + 1], xf[n & 3], acc[q][1]);
      }
    }
    if (!ok) acc[q][0] = acc[q][1] = f32x4{qnan, qnan, qnan, qnan};
  }
  if (e != 0) {
    u32x4 pk[4];
#pragma unroll
    for (int q2 = 0; q2 < 4; ++q2) pk[q2] = mark(pack8(acc[q2][0], acc[q2][1]));
    asm volatile("" : "+v"(pk[0]), "+v"(pk[1]), "+v"(pk[2]), "+v"(pk[3]));
    if (!sent) {
      wait_until(tsend);
#pragma unroll
      for (int q2 = 0; q2 < 3; ++q2) store16(pk[q2], peer, lane * 16 + q2 * 1024, cbase, SYS);
    }
    store16(pk[3], peer, lane * 16 + 3 * 1024, cbase, SYS);
  } else {
    u32x4 pp[4];
    unsigned tm;
    bool ok2 = true;
    for (unsigned it = 1;; ++it) {
      asm volatile("" ::: "memory");
#pragma unroll
      for (int q = 0; q < 4; ++q) pp[q] = __builtin_amdgcn_raw_buffer_load_b128(my, lane * 16, cbase + SLOT + q * 1024, SYS);
      tm = __builtin_amdgcn_raw_buffer_load_b32(my, 0, TMO_OFF, SYS);
      if (__builtin_amdgcn_ballot_w64(umin(umin(pp[0][0], pp[1][0]), umin(pp[2][0], pp[3][0])) == 0) == 0) break;
      if ((it & 63) == 0 && expired(my, call, t0)) { ok2 = false; break; }
      __builtin_amdgcn_s_sleep(2);
    }
    const float poison = (ok2 && tm != call) ? 0.f : qnan;
#pragma unroll
    for (int q = 0; q < 4; ++q) {
      add8(acc[q][0], acc[q][1], pp[q]);
      store16(pack8(silu4(acc[q][0], poison), silu4(acc[q][1], poison)), ro, voff_o,
                                             ((128 * sa + 64 * hb + 16 * q) * 2048 + 1024 * sb + 32 * pu) * 2, 0);
    }
  }
}

extern "C" void kernel_launch(int rank, int nranks, ncclComm_t comm, hipStream_t stream,
                              void* const* d_in, const int* in_sizes, int n_in,
                              void* d_out, int out_size,
                              const int* in_dtypes, int out_dtype,
                              void* const* d_out_all,
                              void* const* d_ws_all, size_t ws_bytes) {
  if (nranks != NRANK || (unsigned)rank >= (unsigned)NRANK || n_in < 2 || ws_bytes < WS_NEED ||
      in_sizes[0] != 2048 * 256 || in_sizes[1] != 256 * 2048 || out_size != 256 * 2048) {
    fprintf(stderr, "unexpected problem geometry\n");
    return;
  }
  Args a;
  a.x = d_in[0];
  a.w = d_in[1];
  a.out = d_out;
  a.ws = d_ws_all[rank];
  const unsigned label = relabel((unsigned)rank);
  for (unsigned b = 0; b < NRANK; ++b) {
    const unsigned e = group_peer((b + XCD0) & 7), d = label ^ e;
    a.grp[b].ws = d_ws_all[relabel(d)];
    a.grp[b].slabs = 0;
    for (unsigned r = 0; r < NRANK; ++r) a.grp[b].slabs |= relabel(d ^ r) << (4 * r);
    a.grp[b].misc = label | e << 4 | ((128 * (d >> 2) * 256 + 128 * (d & 1)) * 2);
    for (unsigned i = 0; i < 4; ++i) a.grp[b].misc |= relabel(d ^ (1 + i + (1 + i >= e))) << (20 + 3 * i);
  }
  hipLaunchKernelGGL(fused_gemm_rs, dim3(256), dim3(512), 0, stream, a);
  hipError_t err = hipGetLastError();
  if (err != hipSuccess) fprintf(stderr, "HIP error %s at launch\n", hipGetErrorString(err));
}
```
